```python
import numpy as np

kernel_name = "gpu_ag_gemm_m4096_k4096_n16384_f32_gelu"


ABSMAX_RTOL = 0.001

COMPUTE_DTYPE = "float32"
EPILOGUE = "gelu"
IN_QUANT = None
OUT_QUANT = None
IN_DTYPES = {"x": "float32", "w_mat": "float32"}
WORLD_SIZE = 8

def setup_inputs():

    x = np.random.randn(4096, 4096).astype(np.float32)
    w_mat = (np.random.randn(4096, 16384) / np.sqrt(4096)).astype(np.float32)
    return {"x": x, "w_mat": w_mat}

def _epilogue(y):
    c = 0.7978845608028654
    return (0.5 * y * (1.0 + np.tanh(c * (y + 0.044715 * y ** 3)))).astype(np.float32)

def reference(x, w_mat):
    y = (x @ w_mat).astype(np.float32)
    return _epilogue(y)
```

```cpp
#include <hip/hip_runtime.h>
#include <rccl/rccl.h>
#include <cstdio>
#include <cstdint>

#define HIP_CHECK(expr) do { hipError_t _e = (expr); if (_e != hipSuccess) { \
    fprintf(stderr, "HIP error %s at %s:%d\n", hipGetErrorString(_e), __FILE__, __LINE__); \
    return; } } while (0)

typedef _Float16 h16x8 __attribute__((ext_vector_type(8)));
typedef float f32x4 __attribute__((ext_vector_type(4)));
typedef unsigned u32x4 __attribute__((ext_vector_type(4)));
#define LAS __attribute__((address_space(3)))

namespace {

constexpr int NR = 8;
constexpr int MSH = 512;
constexpr int KD = 4096;
constexpr int NSH = 2048;
constexpr int NKT = KD / 32;
constexpr int KT_PER_CH = 8;
constexpr int NCH = NKT / KT_PER_CH;
constexpr int NSEND = 32;
constexpr int NGEMM = 256;
constexpr int NWG = NSEND + NGEMM;
constexpr int NT = 512;

constexpr size_t OFF_TICKET = 0;
constexpr size_t OFF_TMO = 128;
constexpr size_t OFF_XCNT = 4096;
constexpr size_t OFF_WCNT = OFF_XCNT + (size_t)NR * NCH * 128;
constexpr size_t OFF_XA = 1u << 20;
constexpr size_t XA_SRC = (size_t)MSH * KD * 2;
constexpr size_t XA_PAR = XA_SRC * NR;
constexpr size_t OFF_WB = OFF_XA + 2 * XA_PAR;
constexpr size_t WB_BYTES = (size_t)KD * NSH * 2;
constexpr size_t WS_NEED = OFF_WB + WB_BYTES;

constexpr int STAGE = 24576;
constexpr int NSTAGE = 3;
constexpr unsigned long long DEADLINE = 100000000ull;

struct Params {
  char* peer[NR];
  const float* x;
  const float* w;
  float* out;
  int rank;
};

__device__ __forceinline__ void wait_ge(const unsigned* p, unsigned target, unsigned* tmo, unsigned call) {
  unsigned long long t0 = __builtin_amdgcn_s_memrealtime();
  while (__hip_atomic_load(p, __ATOMIC_RELAXED, __HIP_MEMORY_SCOPE_SYSTEM) < target) {
    if (__hip_atomic_load(tmo, __ATOMIC_RELAXED, __HIP_MEMORY_SCOPE_AGENT) == call ||
        __builtin_amdgcn_s_memrealtime() - t0 > DEADLINE) {
      __hip_atomic_store(tmo, call, __ATOMIC_RELAXED, __HIP_MEMORY_SCOPE_AGENT);
      break;
    }
    __builtin_amdgcn_s_sleep(1);
  }
  __builtin_amdgcn_fence(__ATOMIC_ACQUIRE, "");
  asm volatile("s_waitcnt vmcnt(0)" ::: "memory");
}

__device__ __forceinline__ u32x4 cvt8(f32x4 a, f32x4 b) {
  h16x8 h;
#pragma unroll
  for (int j = 0; j < 4; ++j) { h[j] = (_Float16)a[j]; h[4 + j] = (_Float16)b[j]; }
  return __builtin_bit_cast(u32x4, h);
}

__device__ __forceinline__ float gelu_tanh(float y) {
  const float c2 = -2.0f * 0.7978845608028654f;
  float u = y * (1.0f + 0.044715f * y * y);
  float e = __expf(c2 * u);
  return y * __builtin_amdgcn_rcpf(1.0f + e);
}

}

__global__ void __launch_bounds__(NT, 4) ag_gemm_gelu_kernel(Params P) {
  __shared__ __attribute__((aligned(16))) char lds_raw[STAGE * NSTAGE + 16];
  LAS char* lds = (LAS char*)lds_raw;
  const int tid = threadIdx.x;
  const int lane = tid & 63;
  const int wave = __builtin_amdgcn_readfirstlane(tid >> 6);
  const int rank = P.rank;
  char* my = P.peer[NR - 1];
  unsigned* tmo = (unsigned*)(my + OFF_TMO);

  LAS unsigned* sh_call = (LAS unsigned*)(lds + STAGE * NSTAGE);
  if (tid == 0) {
    unsigned t = __hip_atomic_fetch_add((unsigned*)(my + OFF_TICKET), 1u, __ATOMIC_RELAXED, __HIP_MEMORY_SCOPE_AGENT);
    *sh_call = t / (unsigned)NWG + 1u;
  }

  if (blockIdx.x < NSEND) {
    const int gw = blockIdx.x * 8 + wave;
    const int ktl = gw >> 5, rb = gw & 31;
    const float* src = P.x + (size_t)(rb * 16 + (lane & 15)) * KD + ktl * 32 + (lane >> 4) * 8;
    char* lane_ws = P.peer[0];
#pragma unroll
    for (int i = 1; i < NR; ++i) lane_ws = (lane == i) ? P.peer[i] : lane_ws;
    unsigned* cnt_dst = (unsigned*)(lane_ws + OFF_XCNT + (size_t)(rank * NCH) * 128);
    f32x4 a0 = *(const f32x4*)(src), a1 = *(const f32x4*)(src + 4);
    __syncthreads();
    const unsigned call = __builtin_amdgcn_readfirstlane(*sh_call);
    const size_t slot = OFF_XA + (size_t)(call & 1u) * XA_PAR + (size_t)rank * XA_SRC;
    __amdgpu_buffer_rsrc_t rs[NR];
#pragma unroll
    for (int p = 0; p < NR; ++p) rs[p] = __builtin_amdgcn_make_buffer_rsrc((void*)(P.peer[p] + slot), 0, (int)XA_SRC, 0x00020000);
    const int voff = ktl * 32768 + rb * 1024 + lane * 16;
    for (int c = 0; c < NCH; ++c) {
      u32x4 u = cvt8(a0, a1);
      if (c + 1 < NCH) { const float* s2 = src + (c + 1) * (KT_PER_CH * 32); a0 = *(const f32x4*)(s2); a1 = *(const f32x4*)(s2 + 4); }
      const int o = voff + c * (KT_PER_CH * 32768);
#pragma unroll
      for (int i = 0; i < NR; ++i) __builtin_amdgcn_raw_buffer_store_b128(u, rs[i], o, 0, 17);
      if (c > 0) {
        asm volatile("s_waitcnt vmcnt(8)" ::: "memory");
        __builtin_amdgcn_s_barrier();
        if (wave == 0 && lane < NR)
          __hip_atomic_fetch_add(cnt_dst + (c - 1) * 32, 1u, __ATOMIC_RELAXED, __HIP_MEMORY_SCOPE_SYSTEM);
      }
    }
    asm volatile("s_waitcnt vmcnt(0)" ::: "memory");
    __builtin_amdgcn_s_barrier();
    if (wave == 0 && lane < NR)
      __hip_atomic_fetch_add(cnt_dst + (NCH - 1) * 32, 1u, __ATOMIC_RELAXED, __HIP_MEMORY_SCOPE_SYSTEM);
    return;
  }

  const int g = blockIdx.x - NSEND;
  const int srcr = g & 7;
  const int idx = g >> 3;
  const int half = idx >> 4, jt = idx & 15;
  const int pi = srcr * 2 + half;

  {
    const int kg = lane >> 4, ng = lane & 15;
    __amdgpu_buffer_rsrc_t rw = __builtin_amdgcn_make_buffer_rsrc((void*)(my + OFF_WB), 0, (int)WB_BYTES, 0x00020000);
#pragma unroll
    for (int nh = 0; nh < 2; ++nh) {
      const float* s = P.w + (size_t)(256 * pi + 32 * wave + 8 * kg) * NSH + 128 * jt + 64 * nh + 4 * ng;
      f32x4 v[8];
#pragma unroll
      for (int j = 0; j < 8; ++j) v[j] = *(const f32x4*)(s + (size_t)j * NSH);
      const int o = jt * (NKT * 8192) + (8 * pi + wave) * 8192 + (4 * nh + (ng >> 2)) * 1024 + (kg * 16 + (ng & 3) * 4) * 16;
#pragma unroll
      for (int i = 0; i < 4; ++i) {
        h16x8 h;
#pragma unroll
        for (int j = 0; j < 8; ++j) h[j] = (_Float16)v[j][i];
        __builtin_amdgcn_raw_buffer_store_b128(__builtin_bit_cast(u32x4, h), rw, o + i * 16, 0, 17);
      }
    }
  }
  asm volatile("s_waitcnt vmcnt(0)" ::: "memory");
  __syncthreads();
  const unsigned call = __builtin_amdgcn_readfirstlane(*sh_call);
  unsigned* wcnt = (unsigned*)(my + OFF_WCNT + (size_t)jt * 128);
  const unsigned* xcnt = (const unsigned*)(my + OFF_XCNT + (size_t)(srcr * NCH) * 128);
  if (tid == 0) {
    __hip_atomic_fetch_add(wcnt, 1u, __ATOMIC_RELAXED, __HIP_MEMORY_SCOPE_AGENT);
    wait_ge(wcnt, call * 16u, tmo, call);
    wait_ge(xcnt, call * (unsigned)NSEND, tmo, call);
  }
  asm volatile("" ::: "memory"); __builtin_amdgcn_s_barrier(); asm volatile("" ::: "memory");

  const char* Ag = my + OFF_XA + (size_t)(call & 1u) * XA_PAR + (size_t)srcr * XA_SRC + half * 16384 + (2 * wave) * 1024 + lane * 16;
  const char* Bg = my + OFF_WB + (size_t)jt * (NKT * 8192) + wave * 1024 + lane * 16;
  auto issue = [&](int kt, int buf) {
    LAS char* d = lds + buf * STAGE;
    __builtin_amdgcn_global_load_lds((const void*)(Ag + (size_t)kt * 32768), (LAS void*)(d + (2 * wave) * 1024), 16, 0, 0);
    __builtin_amdgcn_global_load_lds((const void*)(Ag + (size_t)kt * 32768 + 1024), (LAS void*)(d + (2 * wave + 1) * 1024), 16, 0, 0);
    __builtin_amdgcn_global_load_lds((const void*)(Bg + (size_t)kt * 8192), (LAS void*)(d + 16384 + wave * 1024), 16, 0, 0);
  };

  const int wm = wave & 3, wn = wave >> 2;
  f32x4 acc[4][4];
#pragma unroll
  for (int i = 0; i < 4; ++i)
#pragma unroll
    for (int j = 0; j < 4; ++j) acc[i][j] = (f32x4){0.f, 0.f, 0.f, 0.f};

  issue(0, 0);
  issue(1, 1);
  int buf = 0;
  for (int kt = 0; kt < NKT; ++kt) {
    const int pk = kt + 2;
    if (pk < NKT && (pk & (KT_PER_CH - 1)) == 0) {
      if (tid == 0) wait_ge(xcnt + (pk / KT_PER_CH) * 32, call * (unsigned)NSEND, tmo, call);
    }
    if (kt + 1 < NKT) asm volatile("s_waitcnt vmcnt(3)" ::: "memory");
    else asm volatile("s_waitcnt vmcnt(0)" ::: "memory");
    __builtin_amdgcn_s_barrier();
    asm volatile("" ::: "memory");
    if (pk < NKT) { int pb = buf + 2; if (pb >= NSTAGE) pb -= NSTAGE; issue(pk, pb); }
    const LAS char* sa = lds + buf * STAGE + (wm * 4) * 1024 + lane * 16;
    const LAS char* sb = lds + buf * STAGE + 16384 + (wn * 4) * 1024 + lane * 16;
    h16x8 af[4], bf[4];
#pragma unroll
    for (int i = 0; i < 4; ++i) af[i] = *(const LAS h16x8*)(sa + i * 1024);
#pragma unroll
    for (int j = 0; j < 4; ++j) bf[j] = *(const LAS h16x8*)(sb + j * 1024);
#pragma unroll
    for (int i = 0; i < 4; ++i)
#pragma unroll
      for (int j = 0; j < 4; ++j) acc[i][j] = __builtin_amdgcn_mfma_f32_16x16x32_f16(bf[j], af[i], acc[i][j], 0, 0, 0);
    asm volatile("s_waitcnt lgkmcnt(0)" ::: "memory");
    if (++buf == NSTAGE) buf = 0;
  }

  if (__builtin_amdgcn_readfirstlane(__hip_atomic_load(tmo, __ATOMIC_RELAXED, __HIP_MEMORY_SCOPE_AGENT)) == call) {
    const float q = __builtin_nanf("");
#pragma unroll
    for (int i = 0; i < 4; ++i)
#pragma unroll
      for (int j = 0; j < 4; ++j) acc[i][j] = (f32x4){q, q, q, q};
  }
  float* o = P.out + (size_t)(srcr * MSH + half * 256 + wm * 64 + (lane & 15)) * NSH + jt * 128 + wn * 64 + (lane >> 4) * 4;
#pragma unroll
  for (int i = 0; i < 4; ++i)
#pragma unroll
    for (int j = 0; j < 4; ++j) {
      f32x4 v = acc[i][j], r;
#pragma unroll
      for (int e = 0; e < 4; ++e) r[e] = gelu_tanh(v[e]);
      *(f32x4*)(o + (size_t)(i * 16) * NSH + j * 16) = r;
    }
}

extern "C" void kernel_launch(int rank, int nranks, ncclComm_t comm,
                              hipStream_t stream,
                              void* const* d_in, const int* in_sizes, int n_in,
                              void* d_out, int out_size,
                              const int* in_dtypes, int out_dtype,
                              void* const* d_out_all,
                              void* const* d_ws_all, size_t ws_bytes) {
  (void)comm; (void)in_sizes; (void)n_in; (void)out_size; (void)in_dtypes; (void)out_dtype; (void)d_out_all;
  if (nranks != NR || ws_bytes < WS_NEED || (unsigned)rank >= (unsigned)NR) {
    fprintf(stderr, "unsupported configuration (nranks %d, ws_bytes %zu)\n", nranks, ws_bytes);
    return;
  }
  Params P;
  for (int i = 0; i < NR; ++i) P.peer[i] = (char*)d_ws_all[(rank + 1 + i) % NR];
  P.x = (const float*)d_in[0];
  P.w = (const float*)d_in[1];
  P.out = (float*)d_out;
  P.rank = rank;
  ag_gemm_gelu_kernel<<<NWG, NT, 0, stream>>>(P);
  HIP_CHECK(hipGetLastError());
}
```

```cpp
#include <hip/hip_runtime.h>
#include <rccl/rccl.h>
#include <cstdio>
#include <cstdint>

#define HIP_CHECK(expr) do { hipError_t _e = (expr); if (_e != hipSuccess) { \
    fprintf(stderr, "HIP error %s at %s:%d\n", hipGetErrorString(_e), __FILE__, __LINE__); \
    return; } } while (0)

typedef _Float16 h16x8 __attribute__((ext_vector_type(8)));
typedef float f32x4 __attribute__((ext_vector_type(4)));
typedef unsigned u32x4 __attribute__((ext_vector_type(4)));
#define LAS __attribute__((address_space(3)))

namespace {

constexpr int NR = 8;
constexpr int MSH = 512;
constexpr int KD = 4096;
constexpr int NSH = 2048;
constexpr int NKT = KD / 32;
constexpr int KT_PER_CH = 8;
constexpr int NCH = NKT / KT_PER_CH;
constexpr int NSEND = 32;
constexpr int NGEMM = 224;
constexpr int NWG = NSEND + NGEMM;
constexpr int NT = 512;

constexpr size_t OFF_TICKET = 0;
constexpr size_t OFF_TMO = 128;
constexpr size_t OFF_TK = 256;
constexpr unsigned NBATCH = (4096 + 6) / 7;
constexpr size_t OFF_XCNT = 4096;
constexpr size_t OFF_WCNT = OFF_XCNT + (size_t)NR * NCH * 128;
constexpr size_t OFF_XA = 1u << 20;
constexpr size_t XA_SRC = (size_t)MSH * KD * 2;
constexpr size_t XA_PAR = XA_SRC * NR;
constexpr size_t OFF_WB = OFF_XA + 2 * XA_PAR;
constexpr size_t WB_BYTES = (size_t)KD * NSH * 2;
constexpr size_t WS_NEED = OFF_WB + WB_BYTES;

constexpr int STAGE = 27648;
constexpr int NSTAGE = 5;
constexpr unsigned long long DEADLINE = 100000000ull;

struct Params {
  char* peer[NR];
  const float* x;
  const float* w;
  float* out;
  int rank;
};
#define STAMP(i) do { } while (0)

__device__ __forceinline__ void wait_ge(const unsigned* p, unsigned target, unsigned* tmo, unsigned call) {
  unsigned long long t0 = __builtin_amdgcn_s_memrealtime();
  while (__hip_atomic_load(p, __ATOMIC_RELAXED, __HIP_MEMORY_SCOPE_SYSTEM) < target) {
    if (__hip_atomic_load(tmo, __ATOMIC_RELAXED, __HIP_MEMORY_SCOPE_AGENT) == call ||
        __builtin_amdgcn_s_memrealtime() - t0 > DEADLINE) {
      __hip_atomic_store(tmo, call, __ATOMIC_RELAXED, __HIP_MEMORY_SCOPE_AGENT);
      break;
    }
    __builtin_amdgcn_s_sleep(1);
  }
  asm volatile("" ::: "memory");
}

__device__ __forceinline__ u32x4 cvt8(f32x4 a, f32x4 b) {
  h16x8 h;
#pragma unroll
  for (int j = 0; j < 4; ++j) { h[j] = (_Float16)a[j]; h[4 + j] = (_Float16)b[j]; }
  return __builtin_bit_cast(u32x4, h);
}

__device__ __forceinline__ float gelu_tanh(float y) {
  const float c2 = -2.0f * 0.7978845608028654f;
  float u = y * (1.0f + 0.044715f * y * y);
  float e = __expf(c2 * u);
  return y * __builtin_amdgcn_rcpf(1.0f + e);
}

}

__global__ void __launch_bounds__(NT, 2) ag_gemm_gelu_kernel(Params P) {
  __shared__ __attribute__((aligned(16))) char lds_raw[STAGE * NSTAGE + 16];
  LAS char* lds = (LAS char*)lds_raw;
  const int tid = threadIdx.x;
  const int lane = tid & 63;
  const int wave = __builtin_amdgcn_readfirstlane(tid >> 6);
  const int rank = P.rank;
  char* my = P.peer[NR - 1];
  unsigned* tmo = (unsigned*)(my + OFF_TMO);

  STAMP(0);
  LAS unsigned* sh_call = (LAS unsigned*)(lds + STAGE * NSTAGE);
  if (tid == 0) {
    unsigned t = __hip_atomic_fetch_add((unsigned*)(my + OFF_TICKET), 1u, __ATOMIC_RELAXED, __HIP_MEMORY_SCOPE_AGENT);
    *sh_call = t / (unsigned)NWG + 1u;
  }

  if (blockIdx.x < NSEND) {
    LAS unsigned* ring = (LAS unsigned*)lds;
    __syncthreads();
    const unsigned call = __builtin_amdgcn_readfirstlane(*sh_call);
    unsigned* tk = (unsigned*)(my + OFF_TK + (call & 1u) * 128);
    if (wave == 7) {
      if (lane == 0) {
        if (blockIdx.x == 0) __hip_atomic_store((unsigned*)(my + OFF_TK + ((call + 1u) & 1u) * 128), 0u, __ATOMIC_RELAXED, __HIP_MEMORY_SCOPE_AGENT);
        ring[0] = __hip_atomic_fetch_add(tk, 1u, __ATOMIC_RELAXED, __HIP_MEMORY_SCOPE_AGENT);
        ring[1] = __hip_atomic_fetch_add(tk, 1u, __ATOMIC_RELAXED, __HIP_MEMORY_SCOPE_AGENT);
      }
      char* lane_ws = P.peer[0];
#pragma unroll
      for (int i = 1; i < NR; ++i) lane_ws = ((lane & 7) == i) ? P.peer[i] : lane_ws;
      unsigned* cnt_dst = (unsigned*)(lane_ws + OFF_XCNT + (size_t)(rank * NCH) * 128);
      __builtin_amdgcn_s_waitcnt(0xC07F);
      __builtin_amdgcn_s_barrier();
      unsigned prev = NBATCH;
      for (int i = 0;; ++i) {
        const unsigned cur = __builtin_amdgcn_readfirstlane(ring[i & 3]);
        if (cur < NBATCH) {
          if (lane == 0) ring[(i + 2) & 3] = __hip_atomic_fetch_add(tk, 1u, __ATOMIC_RELAXED, __HIP_MEMORY_SCOPE_AGENT);
        }
        __builtin_amdgcn_s_waitcnt(0xC07F);
        __builtin_amdgcn_s_barrier();
        if (prev < NBATCH) {
          const unsigned f0 = prev * 7u, f1 = f0 + 7u < 4096u ? f0 + 7u : 4096u;
          const unsigned c0 = f0 >> 8, split = (c0 + 1u) << 8;
          const unsigned n0 = (f1 < split ? f1 : split) - f0, n1 = f1 - f0 - n0;
          if (lane < 8) __hip_atomic_fetch_add(cnt_dst + c0 * 32, n0, __ATOMIC_RELAXED, __HIP_MEMORY_SCOPE_SYSTEM);
          else if (lane < 16 && n1) __hip_atomic_fetch_add(cnt_dst + (c0 + 1u) * 32, n1, __ATOMIC_RELAXED, __HIP_MEMORY_SCOPE_SYSTEM);
        }
        if (cur >= NBATCH) break;
        prev = cur;
      }
      return;
    }
    const size_t slot = OFF_XA + (size_t)(call & 1u) * XA_PAR + (size_t)rank * XA_SRC;
    __amdgpu_buffer_rsrc_t rs[NR];
#pragma unroll
    for (int p = 0; p < NR; ++p) rs[p] = __builtin_amdgcn_make_buffer_rsrc((void*)(P.peer[p] + slot), 0, (int)XA_SRC, 0x00020000);
    const float* xl = P.x + (size_t)(lane & 15) * KD + (lane >> 4) * 8;
    auto frag_src = [&](unsigned f) { return xl + (size_t)(f & 31u) * (16 * KD) + (f >> 5) * 32; };
    __builtin_amdgcn_s_barrier();
    unsigned cur = __builtin_amdgcn_readfirstlane(ring[0]);
    unsigned fcur = cur * 7u + (unsigned)wave;
    f32x4 a0 = {}, a1 = {};
    if (cur < NBATCH && fcur < 4096u) { const float* sp = frag_src(fcur); a0 = *(const f32x4*)sp; a1 = *(const f32x4*)(sp + 4); }
    for (int i = 0;; ++i) {
      const unsigned nxt = cur < NBATCH ? __builtin_amdgcn_readfirstlane(ring[(i + 1) & 3]) : NBATCH;
      const unsigned fnxt = nxt * 7u + (unsigned)wave;
      const bool send = cur < NBATCH && fcur < 4096u;
      const u32x4 u = cvt8(a0, a1);
      if (nxt < NBATCH && fnxt < 4096u) { const float* sp = frag_src(fnxt); a0 = *(const f32x4*)sp; a1 = *(const f32x4*)(sp + 4); }
      if (send) {
        const int o = (int)fcur * 1024 + lane * 16;
#pragma unroll
        for (int p = 0; p < NR; ++p) __builtin_amdgcn_raw_buffer_store_b128(u, rs[p], o, 0, 17);
        asm volatile("s_waitcnt vmcnt(8)" ::: "memory");
      } else {
        asm volatile("s_waitcnt vmcnt(0)" ::: "memory");
      }
      __builtin_amdgcn_s_barrier();
      if (cur >= NBATCH) break;
      cur = nxt; fcur = fnxt;
    }
    return;
  }

  const int g = blockIdx.x - NSEND;
  const int jt = ((g & 7) >> 1) * 4 + (g >> 3) / 7;
  const int pi = (g & 1) * 7 + (g >> 3) % 7;
  const int srcr = (rank + 1 + (pi >> 1)) & (NR - 1);
  const int half = pi & 1;
  const int ne = pi < 4 ? 3 : 2;
  const int e0 = pi < 4 ? 3 * pi : 12 + 2 * (pi - 4);

  {
    const int kg = lane >> 4, ng = lane & 15;
    __amdgpu_buffer_rsrc_t rw = __builtin_amdgcn_make_buffer_rsrc((void*)(my + OFF_WB), 0, (int)WB_BYTES, 0x00020000);
    const int gwv = pi * 8 + wave;
    const bool third = gwv + 224 < 256;
    f32x4 v[3][8];
#pragma unroll
    for (int r = 0; r < 3; ++r) {
      const int t = gwv + r * 112;
      if (r < 2 || third) {
        const float* s = P.w + (size_t)(32 * (t >> 1) + 8 * kg) * NSH + 128 * jt + 64 * (t & 1) + 4 * ng;
#pragma unroll
        for (int j = 0; j < 8; ++j) v[r][j] = __builtin_nontemporal_load((const f32x4*)(s + (size_t)j * NSH));
      }
    }
    LAS char* tb = lds + wave * 4096;
#pragma unroll
    for (int r = 0; r < 3; ++r) {
      const int t = gwv + r * 112;
      if (r < 2 || third) {
#pragma unroll
        for (int i = 0; i < 4; ++i) {
          h16x8 h;
#pragma unroll
          for (int j = 0; j < 8; ++j) h[j] = (_Float16)v[r][j][i];
          *(LAS h16x8*)(tb + (ng >> 2) * 1024 + (kg * 16 + (ng & 3) * 4 + i) * 16) = h;
        }
        const int o = jt * (NKT * 8192) + (t >> 1) * 8192 + (4 * (t & 1)) * 1024 + lane * 16;
#pragma unroll
        for (int f = 0; f < 4; ++f) {
          u32x4 q = *(const LAS u32x4*)(tb + f * 1024 + lane * 16);
          __builtin_amdgcn_raw_buffer_store_b128(q, rw, o + f * 1024, 0, 17);
        }
      }
    }
  }
  asm volatile("s_waitcnt vmcnt(0)" ::: "memory");
  __syncthreads();
  STAMP(1);
  const unsigned call = __builtin_amdgcn_readfirstlane(*sh_call);
  unsigned* wcnt = (unsigned*)(my + OFF_WCNT + (size_t)jt * 128);
  const unsigned* xcnt = (const unsigned*)(my + OFF_XCNT + (size_t)(srcr * NCH) * 128);
  const unsigned* lcnt = (const unsigned*)(my + OFF_XCNT + (size_t)(rank * NCH) * 128);
  const unsigned xtarget = call * 256u;
  if (tid == 0) {
    __hip_atomic_fetch_add(wcnt, 1u, __ATOMIC_RELAXED, __HIP_MEMORY_SCOPE_AGENT);
    wait_ge(wcnt, call * 14u, tmo, call);
    STAMP(2);
    wait_ge(lcnt, xtarget, tmo, call);
    wait_ge(xcnt, xtarget, tmo, call);
    STAMP(3);
  }
  asm volatile("" ::: "memory"); __builtin_amdgcn_s_barrier(); asm volatile("" ::: "memory");

  const int grp = wave >> 2, wq = wave & 3, wm = wq & 1, wn = wq >> 1;
  const char* xa = my + OFF_XA + (size_t)(call & 1u) * XA_PAR;
  const bool loader = wq < 3;
  const char* gsrc; int gstride, ldst;
  if (wq < 2) { gsrc = xa + (size_t)srcr * XA_SRC + half * 16384 + wq * 8192; gstride = 32768; ldst = wq * 8192; }
  else        { gsrc = my + OFF_WB + (size_t)jt * (NKT * 8192); gstride = 8192; ldst = 19456; }
  gsrc += lane * 16;
  const int ef = wq < ne ? wq : ne - 1;
  const char* esrc = xa + (size_t)rank * XA_SRC + (e0 + ef) * 1024 + lane * 16;
  auto issue = [&](int kt, int buf) {
    if (loader) {
      LAS char* d = lds + buf * STAGE;
      const char* gp = gsrc + (size_t)kt * gstride;
#pragma unroll
      for (int i = 0; i < 8; ++i)
        __builtin_amdgcn_global_load_lds((const void*)(gp + i * 1024), (LAS void*)(d + ldst + i * 1024), 16, 0, 0);
      __builtin_amdgcn_global_load_lds((const void*)(esrc + (size_t)kt * 32768), (LAS void*)(d + 16384 + ef * 1024), 16, 0, 0);
    }
  };

  f32x4 acc[8][4], acce[2][4];
#pragma unroll
  for (int j = 0; j < 4; ++j) {
    acce[0][j] = (f32x4){0.f, 0.f, 0.f, 0.f}; acce[1][j] = (f32x4){0.f, 0.f, 0.f, 0.f};
#pragma unroll
    for (int i = 0; i < 8; ++i) acc[i][j] = (f32x4){0.f, 0.f, 0.f, 0.f};
  }
  const int eoff0 = 16384 + wm * 1024, eoff1 = 16384 + (wm + 2 < ne ? wm + 2 : 0) * 1024;
  unsigned pre_l = 0, pre_x = 0;
  auto body = [&](int s, bool steady) {
    if (wave == 7) {
      const int pk = s + 5;
      const int ph = pk & (KT_PER_CH - 1);
      if (pk < NKT) {
        if (ph == 0) {
          STAMP(20 + pk / KT_PER_CH);
          if (__builtin_amdgcn_readfirstlane(pre_l) < xtarget) wait_ge(lcnt + (pk / KT_PER_CH) * 32, xtarget, tmo, call);
          if (__builtin_amdgcn_readfirstlane(pre_x) < xtarget) wait_ge(xcnt + (pk / KT_PER_CH) * 32, xtarget, tmo, call);
          STAMP(3 + pk / KT_PER_CH);
        } else if (ph == 4 && (pk | (KT_PER_CH - 1)) + 1 < NKT) {
          pre_l = __hip_atomic_load(lcnt + (pk / KT_PER_CH + 1) * 32, __ATOMIC_RELAXED, __HIP_MEMORY_SCOPE_SYSTEM);
          pre_x = __hip_atomic_load(xcnt + (pk / KT_PER_CH + 1) * 32, __ATOMIC_RELAXED, __HIP_MEMORY_SCOPE_SYSTEM);
        }
      }
    }
    const int rb = s % NSTAGE;
    if (steady) { int ib = rb + 4; if (ib >= NSTAGE) ib -= NSTAGE; issue(s + 4, ib); }
    h16x8 fa[8], fb[4], fe[2];
    {
      const LAS char* st = lds + rb * STAGE + lane * 16;
#pragma unroll
      for (int j = 0; j < 4; ++j) fb[j] = *(const LAS h16x8*)(st + 19456 + (wn * 4 + j) * 1024);
#pragma unroll
      for (int i = 0; i < 8; ++i) fa[i] = *(const LAS h16x8*)(st + (wm * 8 + i) * 1024);
      fe[0] = *(const LAS h16x8*)(st + eoff0); fe[1] = *(const LAS h16x8*)(st + eoff1);
    }
    if (loader) {
      if (steady) asm volatile("s_waitcnt vmcnt(9)" ::: "memory");
      else asm volatile("s_waitcnt vmcnt(0)" ::: "memory");
    }
    __builtin_amdgcn_s_waitcnt(0xC07F);
    __builtin_amdgcn_sched_barrier(0);
    __builtin_amdgcn_s_barrier();
    __builtin_amdgcn_sched_barrier(0);
    __builtin_amdgcn_s_setprio(1);
#pragma unroll
    for (int i = 0; i < 8; ++i)
#pragma unroll
      for (int j = 0; j < 4; ++j) acc[i][j] = __builtin_amdgcn_mfma_f32_16x16x32_f16(fb[j], fa[i], acc[i][j], 0, 0, 0);
#pragma unroll
    for (int r = 0; r < 2; ++r)
#pragma unroll
      for (int j = 0; j < 4; ++j) acce[r][j] = __builtin_amdgcn_mfma_f32_16x16x32_f16(fb[j], fe[r], acce[r][j], 0, 0, 0);
    __builtin_amdgcn_s_setprio(0);
    __builtin_amdgcn_sched_barrier(0);
    __builtin_amdgcn_s_barrier();
    asm volatile("" ::: "memory");
  };
  issue(grp, grp); issue(grp + 2, grp + 2);
  asm volatile("s_waitcnt vmcnt(9)" ::: "memory");
  __builtin_amdgcn_s_barrier();
  asm volatile("" ::: "memory");
  if (grp == 1) __builtin_amdgcn_s_barrier();
  for (int s = grp; s < NKT - 4; s += 2) body(s, true);
  body(NKT - 4 + grp, false);
  body(NKT - 2 + grp, false);
  if (grp == 0) __builtin_amdgcn_s_barrier();
  asm volatile("" ::: "memory");

  STAMP(40);
  f32x4 fin[4][4], fine[4];
  {
    LAS char* mine = lds + wave * 10240 + lane * 16;
    const LAS char* theirs = lds + (wave ^ 4) * 10240 + lane * 16;
#pragma unroll
    for (int rnd = 0; rnd < 2; ++rnd) {
#pragma unroll
      for (int i = 0; i < 2; ++i)
#pragma unroll
        for (int j = 0; j < 4; ++j) {
          f32x4 v = grp == 0 ? acc[4 + 2 * rnd + i][j] : acc[2 * rnd + i][j];
          *(LAS f32x4*)(mine + (i * 4 + j) * 1024) = v;
        }
#pragma unroll
      for (int j = 0; j < 2; ++j) {
        f32x4 v = grp == 0 ? acce[1][2 * rnd + j] : acce[0][2 * rnd + j];
        *(LAS f32x4*)(mine + (8 + j) * 1024) = v;
      }
      __builtin_amdgcn_s_waitcnt(0xC07F);
      __builtin_amdgcn_s_barrier();
      asm volatile("" ::: "memory");
#pragma unroll
      for (int i = 0; i < 2; ++i)
#pragma unroll
        for (int j = 0; j < 4; ++j) {
          f32x4 v = grp == 0 ? acc[2 * rnd + i][j] : acc[4 + 2 * rnd + i][j];
          fin[2 * rnd + i][j] = v + *(const LAS f32x4*)(theirs + (i * 4 + j) * 1024);
        }
#pragma unroll
      for (int j = 0; j < 2; ++j) {
        f32x4 v = grp == 0 ? acce[0][2 * rnd + j] : acce[1][2 * rnd + j];
        fine[2 * rnd + j] = v + *(const LAS f32x4*)(theirs + (8 + j) * 1024);
      }
      if (rnd == 0) { __builtin_amdgcn_s_waitcnt(0xC07F); __builtin_amdgcn_s_barrier(); asm volatile("" ::: "memory"); }
    }
  }
  if (__builtin_amdgcn_readfirstlane(__hip_atomic_load(tmo, __ATOMIC_RELAXED, __HIP_MEMORY_SCOPE_AGENT)) == call) {
    const float q = __builtin_nanf("");
#pragma unroll
    for (int j = 0; j < 4; ++j) {
      fine[j] = (f32x4){q, q, q, q};
#pragma unroll
      for (int i = 0; i < 4; ++i) fin[i][j] = (f32x4){q, q, q, q};
    }
  }
  const int col = jt * 128 + wn * 64 + (lane >> 4) * 4;
  float* o = P.out + (size_t)(srcr * MSH + half * 256 + wm * 128 + grp * 64 + (lane & 15)) * NSH + col;
#pragma unroll
  for (int i = 0; i < 4; ++i)
#pragma unroll
    for (int j = 0; j < 4; ++j) {
      f32x4 v = fin[i][j], r;
#pragma unroll
      for (int e = 0; e < 4; ++e) r[e] = gelu_tanh(v[e]);
      *(f32x4*)(o + (size_t)(i * 16) * NSH + j * 16) = r;
    }
  const int er = wm + 2 * grp;
  if (er < ne) {
    float* oe = P.out + (size_t)(rank * MSH + (e0 + er) * 16 + (lane & 15)) * NSH + col;
#pragma unroll
    for (int j = 0; j < 4; ++j) {
      f32x4 v = fine[j], r;
#pragma unroll
      for (int e = 0; e < 4; ++e) r[e] = gelu_tanh(v[e]);
      *(f32x4*)(oe + j * 16) = r;
    }
  }
}

extern "C" void kernel_launch(int rank, int nranks, ncclComm_t comm,
                              hipStream_t stream,
                              void* const* d_in, const int* in_sizes, int n_in,
                              void* d_out, int out_size,
                              const int* in_dtypes, int out_dtype,
                              void* const* d_out_all,
                              void* const* d_ws_all, size_t ws_bytes) {
  (void)comm; (void)in_sizes; (void)n_in; (void)out_size; (void)in_dtypes; (void)out_dtype; (void)d_out_all;
  if (nranks != NR || ws_bytes < WS_NEED || (unsigned)rank >= (unsigned)NR) {
    fprintf(stderr, "unsupported configuration (nranks %d, ws_bytes %zu)\n", nranks, ws_bytes);
    return;
  }
  Params P;
  for (int i = 0; i < NR; ++i) P.peer[i] = (char*)d_ws_all[(rank + 1 + i) % NR];
  P.x = (const float*)d_in[0];
  P.w = (const float*)d_in[1];
  P.out = (float*)d_out;
  P.rank = rank;
  ag_gemm_gelu_kernel<<<NWG, NT, 0, stream>>>(P);
  HIP_CHECK(hipGetLastError());
}
```

```cpp
#include <hip/hip_runtime.h>
#include <rccl/rccl.h>
#include <cstdio>
#include <cstdint>
#include <type_traits>

#define HIP_CHECK(expr) do { hipError_t _e = (expr); if (_e != hipSuccess) { \
    fprintf(stderr, "HIP error %s at %s:%d\n", hipGetErrorString(_e), __FILE__, __LINE__); \
    return; } } while (0)

typedef _Float16 h16x8 __attribute__((ext_vector_type(8)));
typedef float f32x4 __attribute__((ext_vector_type(4)));
typedef unsigned u32x4 __attribute__((ext_vector_type(4)));
#define LAS __attribute__((address_space(3)))

namespace {

constexpr int NR = 8;
constexpr int MSH = 512;
constexpr int KD = 4096;
constexpr int NSH = 2048;
constexpr int NKT = KD / 32;
#define CH_V 4
#define BS_V 5
constexpr int KT_PER_CH = CH_V;
constexpr unsigned FR_PER_CH = 32u * CH_V;
constexpr unsigned BS = BS_V;
constexpr int NCH = NKT / KT_PER_CH;
#define NSEND_V 32
#define SEND_DEPTH 1
constexpr int NSEND = NSEND_V;
constexpr int NGEMM = 224;
constexpr int NWG = NSEND + NGEMM;
constexpr int NT = 512;

constexpr size_t OFF_TICKET = 0;
constexpr size_t OFF_TMO = 128;
constexpr size_t OFF_TK = 256;
constexpr unsigned NBATCH = (4096 + BS - 1) / BS;
constexpr size_t OFF_XCNT = 4096;
constexpr size_t OFF_WCNT = OFF_XCNT + (size_t)NR * NCH * 128;
constexpr size_t OFF_XA = 1u << 20;
constexpr size_t XA_SRC = (size_t)MSH * KD * 2;
constexpr size_t XA_PAR = XA_SRC * NR;
constexpr size_t OFF_WB = OFF_XA + 2 * XA_PAR;
constexpr size_t WB_BYTES = (size_t)KD * NSH * 2;
constexpr size_t WS_NEED = OFF_WB + WB_BYTES;

constexpr int STAGE = 27648;
constexpr int NSTAGE = 5;
constexpr unsigned long long DEADLINE = 100000000ull;

struct Params {
  char* peer[NR];
  const float* x;
  const float* w;
  float* out;
  int rank;
};
#define STAMP(i) do { } while (0)

__device__ __forceinline__ void wait_ge(const unsigned* p, unsigned target, unsigned* tmo, unsigned call) {
  unsigned long long t0 = __builtin_amdgcn_s_memrealtime();
  while (__hip_atomic_load(p, __ATOMIC_RELAXED, __HIP_MEMORY_SCOPE_SYSTEM) < target) {
    if (__hip_atomic_load(tmo, __ATOMIC_RELAXED, __HIP_MEMORY_SCOPE_AGENT) == call ||
        __builtin_amdgcn_s_memrealtime() - t0 > DEADLINE) {
      __hip_atomic_store(tmo, call, __ATOMIC_RELAXED, __HIP_MEMORY_SCOPE_AGENT);
      break;
    }
    __builtin_amdgcn_s_sleep(1);
  }
  asm volatile("" ::: "memory");
}

__device__ __forceinline__ u32x4 cvt8(f32x4 a, f32x4 b) {
  h16x8 h;
#pragma unroll
  for (int j = 0; j < 4; ++j) { h[j] = (_Float16)a[j]; h[4 + j] = (_Float16)b[j]; }
  return __builtin_bit_cast(u32x4, h);
}

__device__ __forceinline__ float gelu_tanh(float y) {
  constexpr float k1 = -2.0f * 0.7978845608028654f * 1.4426950408889634f;
  constexpr float k2 = k1 * 0.044715f;
  const float e = __builtin_amdgcn_exp2f(y * __builtin_fmaf(k2, y * y, k1));
  return y * __builtin_amdgcn_rcpf(1.0f + e);
}

}

__global__ void __launch_bounds__(NT, 2) ag_gemm_gelu_kernel(Params P) {
  __shared__ __attribute__((aligned(16))) char lds_raw[STAGE * NSTAGE + 16];
  LAS char* lds = (LAS char*)lds_raw;
  const int tid = threadIdx.x;
  const int lane = tid & 63;
  const int wave = __builtin_amdgcn_readfirstlane(tid >> 6);
  const int rank = P.rank;
  char* my = P.peer[NR - 1];
  unsigned* tmo = (unsigned*)(my + OFF_TMO);

  STAMP(0);
  LAS unsigned* sh_call = (LAS unsigned*)(lds + STAGE * NSTAGE);
  if (tid == 0) {
    unsigned t = __hip_atomic_fetch_add((unsigned*)(my + OFF_TICKET), 1u, __ATOMIC_RELAXED, __HIP_MEMORY_SCOPE_AGENT);
    *sh_call = t / (unsigned)NWG + 1u;
  }

  if (blockIdx.x < NSEND) {
    LAS unsigned* ring = (LAS unsigned*)lds;
    const float* xl = P.x + (size_t)(lane & 15) * KD + (lane >> 4) * 8;
    auto frag_src = [&](unsigned f) { return xl + (size_t)(f & 31u) * (16 * KD) + (f >> 5) * 32; };
    unsigned cur = blockIdx.x;
    unsigned fcur = cur * BS + (unsigned)wave;
    f32x4 a0 = {}, a1 = {};
    if (wave < BS) { const float* sp = frag_src(fcur); a0 = *(const f32x4*)sp; a1 = *(const f32x4*)(sp + 4); }
    __syncthreads();
    const unsigned call = __builtin_amdgcn_readfirstlane(*sh_call);
    unsigned* tk = (unsigned*)(my + OFF_TK + (call & 1u) * 128);
    if (wave == 7) {
      if (lane == 0) {
        if (blockIdx.x == 0) __hip_atomic_store((unsigned*)(my + OFF_TK + ((call + 1u) & 1u) * 128), 0u, __ATOMIC_RELAXED, __HIP_MEMORY_SCOPE_AGENT);
        ring[0] = blockIdx.x; ring[1] = NSEND + blockIdx.x;
      }
      char* lane_ws = P.peer[0];
#pragma unroll
      for (int i = 1; i < NR; ++i) lane_ws = ((lane & 7) == i) ? P.peer[i] : lane_ws;
      unsigned* cnt_dst = (unsigned*)(lane_ws + OFF_XCNT + (size_t)(rank * NCH) * 128);
      __builtin_amdgcn_s_waitcnt(0xC07F);
      __builtin_amdgcn_s_barrier();
      unsigned prev = NBATCH;
      for (int i = 0;; ++i) {
        const unsigned tcur = __builtin_amdgcn_readfirstlane(ring[i & 3]);
        if (tcur < NBATCH) {
          if (lane == 0) ring[(i + 2) & 3] = 2u * NSEND + __hip_atomic_fetch_add(tk, 1u, __ATOMIC_RELAXED, __HIP_MEMORY_SCOPE_AGENT);
        }
        __builtin_amdgcn_s_waitcnt(0xC07F);
        __builtin_amdgcn_s_barrier();
        prev = tcur;
        if (prev < NBATCH) {
          const unsigned f0 = prev * BS, f1 = f0 + BS < 4096u ? f0 + BS : 4096u;
          const unsigned c0 = f0 / FR_PER_CH, split = (c0 + 1u) * FR_PER_CH;
          const unsigned n0 = (f1 < split ? f1 : split) - f0, n1 = f1 - f0 - n0;
          if (lane < 8) __hip_atomic_fetch_add(cnt_dst + c0 * 32, n0, __ATOMIC_RELAXED, __HIP_MEMORY_SCOPE_SYSTEM);
          else if (lane < 16 && n1) __hip_atomic_fetch_add(cnt_dst + (c0 + 1u) * 32, n1, __ATOMIC_RELAXED, __HIP_MEMORY_SCOPE_SYSTEM);
        }
        if (tcur >= NBATCH) break;
        prev = tcur;
      }
      return;
    }
    const size_t slot = OFF_XA + (size_t)(call & 1u) * XA_PAR + (size_t)rank * XA_SRC;
    __amdgpu_buffer_rsrc_t rs[NR];
#pragma unroll
    for (int p = 0; p < NR; ++p) rs[p] = __builtin_amdgcn_make_buffer_rsrc((void*)(P.peer[p] + slot), 0, (int)XA_SRC, 0x00020000);
    __builtin_amdgcn_s_barrier();
    for (int i = 0;; ++i) {
      const unsigned nxt = cur < NBATCH ? __builtin_amdgcn_readfirstlane(ring[(i + 1) & 3]) : NBATCH;
      const unsigned fnxt = nxt * BS + (unsigned)wave;
      const bool send = cur < NBATCH && fcur < 4096u && wave < BS;
      const u32x4 u = cvt8(a0, a1);
      if (nxt < NBATCH && fnxt < 4096u && wave < BS) { const float* sp = frag_src(fnxt); a0 = *(const f32x4*)sp; a1 = *(const f32x4*)(sp + 4); }
      if (send) {
        const int o = (int)fcur * 1024 + lane * 16;
#pragma unroll
        for (int p = 0; p < NR; ++p) __builtin_amdgcn_raw_buffer_store_b128(u, rs[p], o, 0, 17);
        asm volatile("s_waitcnt vmcnt(0)" ::: "memory");
      } else {
        asm volatile("s_waitcnt vmcnt(0)" ::: "memory");
      }
      __builtin_amdgcn_s_barrier();
      if (cur >= NBATCH) break;
      cur = nxt; fcur = fnxt;
    }
    return;
  }

  const int g = blockIdx.x - NSEND;
  const int jt = ((g & 7) >> 1) * 4 + (g >> 3) / 7;
  const int pi = (g & 1) * 7 + (g >> 3) % 7;
  const int srcr = (rank + 1 + (pi >> 1)) & (NR - 1);
  const int half = pi & 1;
  const int ne = pi < 4 ? 3 : 2;
  const int e0 = pi < 4 ? 3 * pi : 12 + 2 * (pi - 4);

  {
    const int kg = lane >> 4, ng = lane & 15;
    __amdgpu_buffer_rsrc_t rw = __builtin_amdgcn_make_buffer_rsrc((void*)(my + OFF_WB), 0, (int)WB_BYTES, 0x00020000);
    const int gwv = pi * 8 + wave;
    const bool third = gwv + 224 < 256;
    f32x4 v[3][8];
#pragma unroll
    for (int r = 0; r < 3; ++r) {
      const int t = gwv + r * 112;
      if (r < 2 || third) {
        const float* s = P.w + (size_t)(32 * (t >> 1) + 8 * kg) * NSH + 128 * jt + 64 * (t & 1) + 4 * ng;
#pragma unroll
        for (int j = 0; j < 8; ++j) v[r][j] = __builtin_nontemporal_load((const f32x4*)(s + (size_t)j * NSH));
      }
    }
    LAS char* tb = lds + wave * 4096;
#pragma unroll
    for (int r = 0; r < 3; ++r) {
      const int t = gwv + r * 112;
      if (r < 2 || third) {
#pragma unroll
        for (int i = 0; i < 4; ++i) {
          h16x8 h;
#pragma unroll
          for (int j = 0; j < 8; ++j) h[j] = (_Float16)v[r][j][i];
          *(LAS h16x8*)(tb + (ng >> 2) * 1024 + (kg * 16 + (ng & 3) * 4 + i) * 16) = h;
        }
        const int o = jt * (NKT * 8192) + (t >> 1) * 8192 + (4 * (t & 1)) * 1024 + lane * 16;
#pragma unroll
        for (int f = 0; f < 4; ++f) {
          u32x4 q = *(const LAS u32x4*)(tb + f * 1024 + lane * 16);
          __builtin_amdgcn_raw_buffer_store_b128(q, rw, o + f * 1024, 0, 17);
        }
      }
    }
  }
  asm volatile("s_waitcnt vmcnt(0)" ::: "memory");
  __syncthreads();
  STAMP(1);
  const unsigned call = __builtin_amdgcn_readfirstlane(*sh_call);
  unsigned* wcnt = (unsigned*)(my + OFF_WCNT + (size_t)jt * 128);
  const unsigned* xcnt = (const unsigned*)(my + OFF_XCNT + (size_t)(srcr * NCH) * 128);
  const unsigned* lcnt = (const unsigned*)(my + OFF_XCNT + (size_t)(rank * NCH) * 128);
  const unsigned xtarget = call * FR_PER_CH;
  if (tid == 0) {
    __hip_atomic_fetch_add(wcnt, 1u, __ATOMIC_RELAXED, __HIP_MEMORY_SCOPE_AGENT);
    wait_ge(wcnt, call * 14u, tmo, call);
    STAMP(2);
  }
  asm volatile("" ::: "memory"); __builtin_amdgcn_s_barrier(); asm volatile("" ::: "memory");

  const int grp = wave >> 2, wq = wave & 3, wm = wq & 1, wn = wq >> 1;
  const char* xa = my + OFF_XA + (size_t)(call & 1u) * XA_PAR;
  const bool loader = wq < 3;
  const char* gsrc; int gstride, ldst;
  if (wq < 2) { gsrc = xa + (size_t)srcr * XA_SRC + half * 16384 + wq * 8192; gstride = 32768; ldst = wq * 8192; }
  else        { gsrc = my + OFF_WB + (size_t)jt * (NKT * 8192); gstride = 8192; ldst = 19456; }
  const int ef = wq < ne ? wq : ne - 1;
  const char* esrc = xa + (size_t)rank * XA_SRC + (e0 + ef) * 1024;
  const unsigned l16 = lane * 16;
  auto issue = [&](int kt, int buf) {
    if (loader) {
      LAS char* d = lds + buf * STAGE;
      const char* gp = gsrc + (size_t)kt * gstride;
#pragma unroll
      for (int i = 0; i < 8; ++i)
        __builtin_amdgcn_global_load_lds((const void*)(gp + i * 1024 + l16), (LAS void*)(d + ldst + i * 1024), 16, 0, 0);
      __builtin_amdgcn_global_load_lds((const void*)(esrc + (size_t)kt * 32768 + l16), (LAS void*)(d + 16384 + ef * 1024), 16, 0, 0);
    }
  };

  f32x4 acc[8][4], acce[2][4];
#pragma unroll
  for (int j = 0; j < 4; ++j) {
    acce[0][j] = (f32x4){0.f, 0.f, 0.f, 0.f}; acce[1][j] = (f32x4){0.f, 0.f, 0.f, 0.f};
#pragma unroll
    for (int i = 0; i < 8; ++i) acc[i][j] = (f32x4){0.f, 0.f, 0.f, 0.f};
  }
  const int eoff0 = 16384 + wm * 1024, eoff1 = 16384 + (wm + 2 < ne ? wm + 2 : 0) * 1024;
  LAS int* rdyw = (LAS int*)(lds + STAGE * NSTAGE + 4);
  auto body = [&](int s, bool steady, int probe) {
    if (wave == 7 && probe != 0) {
      if (probe < 0) {
        if (lane == 0) {
          __builtin_amdgcn_global_load_lds((const void*)(lcnt - probe * 32), (LAS void*)(rdyw + 1), 4, 0, 17);
          __builtin_amdgcn_global_load_lds((const void*)(xcnt - probe * 32), (LAS void*)(rdyw + 2), 4, 0, 17);
        }
      } else {
        asm volatile("s_waitcnt vmcnt(0)" ::: "memory");
        const unsigned vl = __builtin_amdgcn_readfirstlane((unsigned)rdyw[1]), vx = __builtin_amdgcn_readfirstlane((unsigned)rdyw[2]);
        bool ok = vl >= xtarget && vx >= xtarget;
        if (!ok && probe < NCH - 1) {
          STAMP(20 + (probe & 15));
          wait_ge(lcnt + probe * 32, xtarget, tmo, call); wait_ge(xcnt + probe * 32, xtarget, tmo, call);
          STAMP(3 + (probe & 15));
          ok = true;
        }
        if (lane == 0) *rdyw = ok ? probe : 0;
      }
    }
    const int rb = s % NSTAGE;
    if (steady) { int ib = rb + 4; if (ib >= NSTAGE) ib -= NSTAGE; issue(s + 4, ib); }
    h16x8 fa[8], fb[4], fe[2];
    {
      const LAS char* st = lds + rb * STAGE + lane * 16;
#pragma unroll
      for (int j = 0; j < 4; ++j) fb[j] = *(const LAS h16x8*)(st + 19456 + (wn * 4 + j) * 1024);
#pragma unroll
      for (int i = 0; i < 8; ++i) fa[i] = *(const LAS h16x8*)(st + (wm * 8 + i) * 1024);
      fe[0] = *(const LAS h16x8*)(st + eoff0); fe[1] = *(const LAS h16x8*)(st + eoff1);
    }
    if (loader) {
      if (steady) asm volatile("s_waitcnt vmcnt(9)" ::: "memory");
      else asm volatile("s_waitcnt vmcnt(0)" ::: "memory");
    }
    __builtin_amdgcn_s_waitcnt(0xC07F);
    __builtin_amdgcn_sched_barrier(0);
    __builtin_amdgcn_s_barrier();
    __builtin_amdgcn_sched_barrier(0);
    __builtin_amdgcn_s_setprio(1);
#pragma unroll
    for (int i = 0; i < 8; ++i)
#pragma unroll
      for (int j = 0; j < 4; ++j) acc[i][j] = __builtin_amdgcn_mfma_f32_16x16x32_f16(fb[j], fa[i], acc[i][j], 0, 0, 0);
#pragma unroll
    for (int r = 0; r < 2; ++r)
#pragma unroll
      for (int j = 0; j < 4; ++j) acce[r][j] = __builtin_amdgcn_mfma_f32_16x16x32_f16(fb[j], fe[r], acce[r][j], 0, 0, 0);
    __builtin_amdgcn_s_setprio(0);
    __builtin_amdgcn_sched_barrier(0);
    __builtin_amdgcn_s_barrier();
    asm volatile("" ::: "memory");
  };
  auto start = [&](int c) {
    const int s0 = c * KT_PER_CH + grp;
    if (wave == 7) {
      STAMP(20 + (c & 15));
      wait_ge(lcnt + c * 32, xtarget, tmo, call); wait_ge(xcnt + c * 32, xtarget, tmo, call);
      if (KT_PER_CH < 8 && c + 1 < NCH - 1) { wait_ge(lcnt + (c + 1) * 32, xtarget, tmo, call); wait_ge(xcnt + (c + 1) * 32, xtarget, tmo, call); }
      STAMP(3 + (c & 15));
    }
    __builtin_amdgcn_s_barrier();
    asm volatile("" ::: "memory");
    issue(s0, s0 % NSTAGE); issue(s0 + 2, (s0 + 2) % NSTAGE);
    asm volatile("s_waitcnt vmcnt(9)" ::: "memory");
    __builtin_amdgcn_s_barrier();
    asm volatile("" ::: "memory");
    if (grp == 1) __builtin_amdgcn_s_barrier();
  };
  constexpr int LAST = NKT - KT_PER_CH;
  bool cont = false;
#pragma unroll 1
  for (int pass = 0; pass < 2; ++pass) {
    if (!cont) {
      if (pass == 1 && grp == 0) __builtin_amdgcn_s_barrier();
      asm volatile("" ::: "memory");
      start(pass == 0 ? 0 : NCH - 1);
      cont = true;
    }
    const int s1 = pass == 0 ? LAST : NKT;
#pragma unroll 1
    for (int s = (pass == 0 ? 0 : LAST) + grp; s < s1; s += 2) {
      if (s == LAST - 4 + grp) cont = __builtin_amdgcn_readfirstlane(*rdyw) == NCH - 1;
      int probe = 0;
      if ((s + 5) % KT_PER_CH == 0 && s + 5 < NKT) probe = (s + 5) / KT_PER_CH;
      else if ((s + 7) % KT_PER_CH == 0 && s + 7 < NKT) probe = -((s + 7) / KT_PER_CH);
      body(s, s + 4 < NKT && (s + 4 < LAST || cont), probe);
    }
  }
  if (grp == 0) __builtin_amdgcn_s_barrier();
  asm volatile("" ::: "memory");

  STAMP(40);
  const bool bad = __builtin_amdgcn_readfirstlane(__hip_atomic_load(tmo, __ATOMIC_RELAXED, __HIP_MEMORY_SCOPE_AGENT)) == call;
  const float poison = bad ? __builtin_nanf("") : 0.0f;
  const int col = jt * 128 + wn * 64 + (lane >> 4) * 4;
  float* o = P.out + (size_t)(srcr * MSH + half * 256 + wm * 128 + grp * 64 + (lane & 15)) * NSH + col;
  const int er = wm + 2 * grp;
  float* oe = P.out + (size_t)(rank * MSH + (e0 + (er < ne ? er : 0)) * 16 + (lane & 15)) * NSH + col;
  auto put = [&](float* dst, f32x4 v) {
    f32x4 r;
#pragma unroll
    for (int e = 0; e < 4; ++e) r[e] = gelu_tanh(v[e] + poison);
    *(f32x4*)dst = r;
  };
  {
    auto finish = [&](auto G) {
      constexpr int g = decltype(G)::value;
      LAS char* mine = lds + wave * 16384 + lane * 16;
      const LAS char* theirs = lds + (wave ^ 4) * 16384 + lane * 16;
#pragma unroll
      for (int q = 0; q < 3; ++q) {
#pragma unroll
        for (int j = 0; j < 4; ++j) *(LAS f32x4*)(mine + (q * 5 + j) * 1024) = acc[(1 - g) * 4 + q][j];
        *(LAS f32x4*)(mine + (q * 5 + 4) * 1024) = acce[1 - g][q];
      }
      __builtin_amdgcn_s_waitcnt(0xC07F);
      __builtin_amdgcn_s_barrier();
      asm volatile("" ::: "memory");
#pragma unroll
      for (int q = 0; q < 3; ++q) {
#pragma unroll
        for (int j = 0; j < 4; ++j) put(o + (size_t)(q * 16) * NSH + j * 16, acc[g * 4 + q][j] + *(const LAS f32x4*)(theirs + (q * 5 + j) * 1024));
        const f32x4 v = acce[g][q] + *(const LAS f32x4*)(theirs + (q * 5 + 4) * 1024);
        if (er < ne) put(oe + q * 16, v);
      }
      __builtin_amdgcn_s_waitcnt(0xC07F);
      __builtin_amdgcn_s_barrier();
      asm volatile("" ::: "memory");
#pragma unroll
      for (int j = 0; j < 4; ++j) *(LAS f32x4*)(mine + j * 1024) = acc[(1 - g) * 4 + 3][j];
      *(LAS f32x4*)(mine + 4 * 1024) = acce[1 - g][3];
      __builtin_amdgcn_s_waitcnt(0xC07F);
      __builtin_amdgcn_s_barrier();
      asm volatile("" ::: "memory");
#pragma unroll
      for (int j = 0; j < 4; ++j) put(o + (size_t)(3 * 16) * NSH + j * 16, acc[g * 4 + 3][j] + *(const LAS f32x4*)(theirs + j * 1024));
      const f32x4 v = acce[g][3] + *(const LAS f32x4*)(theirs + 4 * 1024);
      if (er < ne) put(oe + 3 * 16, v);
    };
    if (grp == 0) finish(std::integral_constant<int, 0>{}); else finish(std::integral_constant<int, 1>{});
  }
}

extern "C" void kernel_launch(int rank, int nranks, ncclComm_t comm,
                              hipStream_t stream,
                              void* const* d_in, const int* in_sizes, int n_in,
                              void* d_out, int out_size,
                              const int* in_dtypes, int out_dtype,
                              void* const* d_out_all,
                              void* const* d_ws_all, size_t ws_bytes) {
  (void)comm; (void)in_sizes; (void)n_in; (void)out_size; (void)in_dtypes; (void)out_dtype; (void)d_out_all;
  if (nranks != NR || ws_bytes < WS_NEED || (unsigned)rank >= (unsigned)NR) {
    fprintf(stderr, "unsupported configuration (nranks %d, ws_bytes %zu)\n", nranks, ws_bytes);
    return;
  }
  Params P;
  for (int i = 0; i < NR; ++i) P.peer[i] = (char*)d_ws_all[(rank + 1 + i) % NR];
  P.x = (const float*)d_in[0];
  P.w = (const float*)d_in[1];
  P.out = (float*)d_out;
  P.rank = rank;
  ag_gemm_gelu_kernel<<<NWG, NT, 0, stream>>>(P);
  HIP_CHECK(hipGetLastError());
}
```

```cpp
#include <hip/hip_runtime.h>
#include <rccl/rccl.h>
#include <cstdio>
#include <cstdint>
#include <type_traits>

#define HIP_CHECK(expr) do { hipError_t _e = (expr); if (_e != hipSuccess) { \
    fprintf(stderr, "HIP error %s at %s:%d\n", hipGetErrorString(_e), __FILE__, __LINE__); \
    return; } } while (0)

typedef _Float16 h16x8 __attribute__((ext_vector_type(8)));
typedef float f32x4 __attribute__((ext_vector_type(4)));
typedef unsigned u32x4 __attribute__((ext_vector_type(4)));
#define LAS __attribute__((address_space(3)))

namespace {

constexpr int NR = 8;
constexpr int MSH = 512;
constexpr int KD = 4096;
constexpr int NSH = 2048;
constexpr int NKT = KD / 32;
constexpr int KT_PER_CH = 4;
constexpr unsigned FR_PER_CH = 32u * KT_PER_CH;
constexpr unsigned BS = 5;
constexpr int NCH = NKT / KT_PER_CH;
constexpr int NSEND = 32;
constexpr int NGEMM = 224;
constexpr int NWG = NSEND + NGEMM;
constexpr int NT = 512;

constexpr size_t OFF_TICKET = 0;
constexpr size_t OFF_TMO = 128;
constexpr size_t OFF_TK = 256;
constexpr unsigned NBATCH = (4096 + BS - 1) / BS;
constexpr size_t OFF_XCNT = 4096;
constexpr size_t OFF_WCNT = OFF_XCNT + (size_t)NR * NCH * 128;
constexpr size_t OFF_XA = 1u << 20;
constexpr size_t XA_SRC = (size_t)MSH * KD * 2;
constexpr size_t XA_PAR = XA_SRC * NR;
constexpr size_t OFF_WB = OFF_XA + 2 * XA_PAR;
constexpr size_t WB_BYTES = (size_t)KD * NSH * 2;
constexpr size_t WS_NEED = OFF_WB + WB_BYTES;

constexpr int STAGE = 27648;
constexpr int NSTAGE = 5;
constexpr unsigned long long DEADLINE = 100000000ull;

struct Params {
  char* peer[NR];
  const float* x;
  const float* w;
  float* out;
  int rank;
};

__device__ __forceinline__ void wait_ge(const unsigned* p, unsigned target, unsigned* tmo, unsigned call) {
  unsigned long long t0 = __builtin_amdgcn_s_memrealtime();
  while (__hip_atomic_load(p, __ATOMIC_RELAXED, __HIP_MEMORY_SCOPE_SYSTEM) < target) {
    if (__hip_atomic_load(tmo, __ATOMIC_RELAXED, __HIP_MEMORY_SCOPE_AGENT) == call ||
        __builtin_amdgcn_s_memrealtime() - t0 > DEADLINE) {
      __hip_atomic_store(tmo, call, __ATOMIC_RELAXED, __HIP_MEMORY_SCOPE_AGENT);
      break;
    }
    __builtin_amdgcn_s_sleep(1);
  }
  asm volatile("" ::: "memory");
}

__device__ __forceinline__ u32x4 cvt8(f32x4 a, f32x4 b) {
  h16x8 h;
#pragma unroll
  for (int j = 0; j < 4; ++j) { h[j] = (_Float16)a[j]; h[4 + j] = (_Float16)b[j]; }
  return __builtin_bit_cast(u32x4, h);
}

__device__ __forceinline__ float gelu_tanh(float y) {
  constexpr float k1 = -2.0f * 0.7978845608028654f * 1.4426950408889634f;
  constexpr float k2 = k1 * 0.044715f;
  const float e = __builtin_amdgcn_exp2f(y * __builtin_fmaf(k2, y * y, k1));
  return y * __builtin_amdgcn_rcpf(1.0f + e);
}

}

__global__ void __launch_bounds__(NT, 2) ag_gemm_gelu_kernel(Params P) {
  __shared__ __attribute__((aligned(16))) char lds_raw[STAGE * NSTAGE + 16];
  LAS char* lds = (LAS char*)lds_raw;
  const int tid = threadIdx.x;
  const int lane = tid & 63;
  const int wave = __builtin_amdgcn_readfirstlane(tid >> 6);
  const int rank = P.rank;
  char* my = P.peer[NR - 1];
  unsigned* tmo = (unsigned*)(my + OFF_TMO);

  LAS unsigned* sh_call = (LAS unsigned*)(lds + STAGE * NSTAGE);
  if (tid == 0) {
    unsigned t = __hip_atomic_fetch_add((unsigned*)(my + OFF_TICKET), 1u, __ATOMIC_RELAXED, __HIP_MEMORY_SCOPE_AGENT);
    *sh_call = t / (unsigned)NWG + 1u;
  }

  if (blockIdx.x < NSEND) {
    LAS unsigned* ring = (LAS unsigned*)lds;
    const float* xl = P.x + (size_t)(lane & 15) * KD + (lane >> 4) * 8;
    auto frag_src = [&](unsigned f) { return xl + (size_t)(f & 31u) * (16 * KD) + (f >> 5) * 32; };
    unsigned cur = blockIdx.x;
    unsigned fcur = cur * BS + (unsigned)wave;
    f32x4 a0 = {}, a1 = {};
    if (wave < BS) { const float* sp = frag_src(fcur); a0 = *(const f32x4*)sp; a1 = *(const f32x4*)(sp + 4); }
    __syncthreads();
    const unsigned call = __builtin_amdgcn_readfirstlane(*sh_call);
    unsigned* tk = (unsigned*)(my + OFF_TK + (call & 1u) * 128);
    if (wave == 7) {
      if (lane == 0) {
        if (blockIdx.x == 0) __hip_atomic_store((unsigned*)(my + OFF_TK + ((call + 1u) & 1u) * 128), 0u, __ATOMIC_RELAXED, __HIP_MEMORY_SCOPE_AGENT);
        ring[0] = blockIdx.x; ring[1] = NSEND + blockIdx.x;
      }
      char* lane_ws = P.peer[0];
#pragma unroll
      for (int i = 1; i < NR; ++i) lane_ws = ((lane & 7) == i) ? P.peer[i] : lane_ws;
      unsigned* cnt_dst = (unsigned*)(lane_ws + OFF_XCNT + (size_t)(rank * NCH) * 128);
      __builtin_amdgcn_s_waitcnt(0xC07F);
      __builtin_amdgcn_s_barrier();
      unsigned prev = NBATCH;
      for (int i = 0;; ++i) {
        const unsigned tcur = __builtin_amdgcn_readfirstlane(ring[i & 3]);
        if (tcur < NBATCH) {
          if (lane == 0) ring[(i + 2) & 3] = 2u * NSEND + __hip_atomic_fetch_add(tk, 1u, __ATOMIC_RELAXED, __HIP_MEMORY_SCOPE_AGENT);
        }
        __builtin_amdgcn_s_waitcnt(0xC07F);
        __builtin_amdgcn_s_barrier();
        prev = tcur;
        if (prev < NBATCH) {
          const unsigned f0 = prev * BS, f1 = f0 + BS < 4096u ? f0 + BS : 4096u;
          const unsigned c0 = f0 / FR_PER_CH, split = (c0 + 1u) * FR_PER_CH;
          const unsigned n0 = (f1 < split ? f1 : split) - f0, n1 = f1 - f0 - n0;
          if (lane < 8) __hip_atomic_fetch_add(cnt_dst + c0 * 32, n0, __ATOMIC_RELAXED, __HIP_MEMORY_SCOPE_SYSTEM);
          else if (lane < 16 && n1) __hip_atomic_fetch_add(cnt_dst + (c0 + 1u) * 32, n1, __ATOMIC_RELAXED, __HIP_MEMORY_SCOPE_SYSTEM);
        }
        if (tcur >= NBATCH) break;
      }
      return;
    }
    const size_t slot = OFF_XA + (size_t)(call & 1u) * XA_PAR + (size_t)rank * XA_SRC;
    __amdgpu_buffer_rsrc_t rs[NR];
#pragma unroll
    for (int p = 0; p < NR; ++p) rs[p] = __builtin_amdgcn_make_buffer_rsrc((void*)(P.peer[p] + slot), 0, (int)XA_SRC, 0x00020000);
    __builtin_amdgcn_s_barrier();
    for (int i = 0;; ++i) {
      const unsigned nxt = cur < NBATCH ? __builtin_amdgcn_readfirstlane(ring[(i + 1) & 3]) : NBATCH;
      const unsigned fnxt = nxt * BS + (unsigned)wave;
      const bool send = cur < NBATCH && fcur < 4096u && wave < BS;
      const u32x4 u = cvt8(a0, a1);
      if (nxt < NBATCH && fnxt < 4096u && wave < BS) { const float* sp = frag_src(fnxt); a0 = *(const f32x4*)sp; a1 = *(const f32x4*)(sp + 4); }
      if (send) {
        const int o = (int)fcur * 1024 + lane * 16;
#pragma unroll
        for (int p = 0; p < NR; ++p) __builtin_amdgcn_raw_buffer_store_b128(u, rs[p], o, 0, 17);
      }
      asm volatile("s_waitcnt vmcnt(0)" ::: "memory");
      __builtin_amdgcn_s_barrier();
      if (cur >= NBATCH) break;
      cur = nxt; fcur = fnxt;
    }
    return;
  }

  const int g = blockIdx.x - NSEND;
  const int jt = ((g & 7) >> 1) * 4 + (g >> 3) / 7;
  const int pi = (g & 1) * 7 + (g >> 3) % 7;
  const int srcr = (rank + 1 + (pi >> 1)) & (NR - 1);
  const int half = pi & 1;
  const int ne = pi < 4 ? 3 : 2;
  const int e0 = pi < 4 ? 3 * pi : 12 + 2 * (pi - 4);

  __builtin_amdgcn_s_sleep(80);

  {
    const int kg = lane >> 4, ng = lane & 15;
    __amdgpu_buffer_rsrc_t rw = __builtin_amdgcn_make_buffer_rsrc((void*)(my + OFF_WB), 0, (int)WB_BYTES, 0x00020000);
    const int gwv = pi * 8 + wave;
    const bool third = gwv + 224 < 256;
    f32x4 v[3][8];
#pragma unroll
    for (int r = 0; r < 3; ++r) {
      const int t = gwv + r * 112;
      if (r < 2 || third) {
        const float* s = P.w + (size_t)(32 * (t >> 1) + 8 * kg) * NSH + 128 * jt + 64 * (t & 1) + 4 * ng;
#pragma unroll
        for (int j = 0; j < 8; ++j) v[r][j] = __builtin_nontemporal_load((const f32x4*)(s + (size_t)j * NSH));
      }
    }
    LAS char* tb = lds + wave * 4096;
#pragma unroll
    for (int r = 0; r < 3; ++r) {
      const int t = gwv + r * 112;
      if (r < 2 || third) {
#pragma unroll
        for (int i = 0; i < 4; ++i) {
          h16x8 h;
#pragma unroll
          for (int j = 0; j < 8; ++j) h[j] = (_Float16)v[r][j][i];
          *(LAS h16x8*)(tb + (ng >> 2) * 1024 + (kg * 16 + (ng & 3) * 4 + i) * 16) = h;
        }
        const int o = jt * (NKT * 8192) + (t >> 1) * 8192 + (4 * (t & 1)) * 1024 + lane * 16;
#pragma unroll
        for (int f = 0; f < 4; ++f) {
          u32x4 q = *(const LAS u32x4*)(tb + f * 1024 + lane * 16);
          __builtin_amdgcn_raw_buffer_store_b128(q, rw, o + f * 1024, 0, 17);
        }
      }
    }
  }
  asm volatile("s_waitcnt vmcnt(0)" ::: "memory");
  __syncthreads();
  const unsigned call = __builtin_amdgcn_readfirstlane(*sh_call);
  unsigned* wcnt = (unsigned*)(my + OFF_WCNT + (size_t)jt * 128);
  const unsigned* xcnt = (const unsigned*)(my + OFF_XCNT + (size_t)(srcr * NCH) * 128);
  const unsigned* lcnt = (const unsigned*)(my + OFF_XCNT + (size_t)(rank * NCH) * 128);
  const unsigned xtarget = call * FR_PER_CH;
  if (tid == 0) {
    __hip_atomic_fetch_add(wcnt, 1u, __ATOMIC_RELAXED, __HIP_MEMORY_SCOPE_AGENT);
    wait_ge(wcnt, call * 14u, tmo, call);
  }
  asm volatile("" ::: "memory"); __builtin_amdgcn_s_barrier(); asm volatile("" ::: "memory");

  const int grp = wave >> 2, wq = wave & 3, wm = wq & 1, wn = wq >> 1;
  const char* xa = my + OFF_XA + (size_t)(call & 1u) * XA_PAR;
  const bool loader = wq < 3;
  const char* gsrc; int gstride, ldst;
  if (wq < 2) { gsrc = xa + (size_t)srcr * XA_SRC + half * 16384 + wq * 8192; gstride = 32768; ldst = wq * 8192; }
  else        { gsrc = my + OFF_WB + (size_t)jt * (NKT * 8192); gstride = 8192; ldst = 19456; }
  const int ef = wq < ne ? wq : ne - 1;
  const char* esrc = xa + (size_t)rank * XA_SRC + (e0 + ef) * 1024;
  const unsigned l16 = lane * 16;
  auto issue = [&](int kt, int buf) {
    if (loader) {
      LAS char* d = lds + buf * STAGE;
      const char* gp = gsrc + (size_t)kt * gstride;
#pragma unroll
      for (int i = 0; i < 8; ++i)
        __builtin_amdgcn_global_load_lds((const void*)(gp + i * 1024 + l16), (LAS void*)(d + ldst + i * 1024), 16, 0, 0);
      __builtin_amdgcn_global_load_lds((const void*)(esrc + (size_t)kt * 32768 + l16), (LAS void*)(d + 16384 + ef * 1024), 16, 0, 0);
    }
  };

  f32x4 acc[8][4], acce[2][4];
#pragma unroll
  for (int j = 0; j < 4; ++j) {
    acce[0][j] = (f32x4){0.f, 0.f, 0.f, 0.f}; acce[1][j] = (f32x4){0.f, 0.f, 0.f, 0.f};
#pragma unroll
    for (int i = 0; i < 8; ++i) acc[i][j] = (f32x4){0.f, 0.f, 0.f, 0.f};
  }
  const int eoff0 = 16384 + wm * 1024, eoff1 = 16384 + (wm + 2 < ne ? wm + 2 : 0) * 1024;
  LAS int* rdyw = (LAS int*)(lds + STAGE * NSTAGE + 4);
  auto body = [&](int s, bool steady, int probe) {
    if (wave == 7 && probe != 0) {
      if (probe < 0) {
        if (lane == 0) {
          __builtin_amdgcn_global_load_lds((const void*)(lcnt - probe * 32), (LAS void*)(rdyw + 1), 4, 0, 17);
          __builtin_amdgcn_global_load_lds((const void*)(xcnt - probe * 32), (LAS void*)(rdyw + 2), 4, 0, 17);
        }
      } else {
        asm volatile("s_waitcnt vmcnt(0)" ::: "memory");
        const unsigned vl = __builtin_amdgcn_readfirstlane((unsigned)rdyw[1]), vx = __builtin_amdgcn_readfirstlane((unsigned)rdyw[2]);
        bool ok = vl >= xtarget && vx >= xtarget;
        if (!ok && probe < NCH - 1) {
          wait_ge(lcnt + probe * 32, xtarget, tmo, call); wait_ge(xcnt + probe * 32, xtarget, tmo, call);
          ok = true;
        }
        if (lane == 0) *rdyw = ok ? probe : 0;
      }
    }
    const int rb = s % NSTAGE;
    if (steady) { int ib = rb + 4; if (ib >= NSTAGE) ib -= NSTAGE; issue(s + 4, ib); }
    h16x8 fa[8], fb[4], fe[2];
    {
      const LAS char* st = lds + rb * STAGE + lane * 16;
#pragma unroll
      for (int j = 0; j < 4; ++j) fb[j] = *(const LAS h16x8*)(st + 19456 + (wn * 4 + j) * 1024);
#pragma unroll
      for (int i = 0; i < 8; ++i) fa[i] = *(const LAS h16x8*)(st + (wm * 8 + i) * 1024);
      fe[0] = *(const LAS h16x8*)(st + eoff0); fe[1] = *(const LAS h16x8*)(st + eoff1);
    }
    if (loader) {
      if (steady) asm volatile("s_waitcnt vmcnt(9)" ::: "memory");
      else asm volatile("s_waitcnt vmcnt(0)" ::: "memory");
    }
    __builtin_amdgcn_s_waitcnt(0xC07F);
    __builtin_amdgcn_sched_barrier(0);
    __builtin_amdgcn_s_barrier();
    __builtin_amdgcn_sched_barrier(0);
    __builtin_amdgcn_s_setprio(1);
#pragma unroll
    for (int i = 0; i < 8; ++i)
#pragma unroll
      for (int j = 0; j < 4; ++j) acc[i][j] = __builtin_amdgcn_mfma_f32_16x16x32_f16(fb[j], fa[i], acc[i][j], 0, 0, 0);
#pragma unroll
    for (int r = 0; r < 2; ++r)
#pragma unroll
      for (int j = 0; j < 4; ++j) acce[r][j] = __builtin_amdgcn_mfma_f32_16x16x32_f16(fb[j], fe[r], acce[r][j], 0, 0, 0);
    __builtin_amdgcn_s_setprio(0);
    __builtin_amdgcn_sched_barrier(0);
    __builtin_amdgcn_s_barrier();
    asm volatile("" ::: "memory");
  };
  auto start = [&](int c) {
    const int s0 = c * KT_PER_CH + grp;
    if (wave == 7) {
      wait_ge(lcnt + c * 32, xtarget, tmo, call); wait_ge(xcnt + c * 32, xtarget, tmo, call);
      if (KT_PER_CH < 8 && c + 1 < NCH - 1) { wait_ge(lcnt + (c + 1) * 32, xtarget, tmo, call); wait_ge(xcnt + (c + 1) * 32, xtarget, tmo, call); }
    }
    __builtin_amdgcn_s_barrier();
    asm volatile("" ::: "memory");
    issue(s0, s0 % NSTAGE); issue(s0 + 2, (s0 + 2) % NSTAGE);
    asm volatile("s_waitcnt vmcnt(9)" ::: "memory");
    __builtin_amdgcn_s_barrier();
    asm volatile("" ::: "memory");
    if (grp == 1) __builtin_amdgcn_s_barrier();
  };
  constexpr int LAST = NKT - KT_PER_CH;
  bool cont = false;
#pragma unroll 1
  for (int pass = 0; pass < 2; ++pass) {
    if (!cont) {
      if (pass == 1 && grp == 0) __builtin_amdgcn_s_barrier();
      asm volatile("" ::: "memory");
      start(pass == 0 ? 0 : NCH - 1);
      cont = true;
    }
    const int s1 = pass == 0 ? LAST : NKT;
#pragma unroll 1
    for (int s = (pass == 0 ? 0 : LAST) + grp; s < s1; s += 2) {
      if (s == LAST - 4 + grp) cont = __builtin_amdgcn_readfirstlane(*rdyw) == NCH - 1;
      int probe = 0;
      if ((s + 5) % KT_PER_CH == 0 && s + 5 < NKT) probe = (s + 5) / KT_PER_CH;
      else if ((s + 7) % KT_PER_CH == 0 && s + 7 < NKT) probe = -((s + 7) / KT_PER_CH);
      body(s, s + 4 < NKT && (s + 4 < LAST || cont), probe);
    }
  }
  if (grp == 0) __builtin_amdgcn_s_barrier();
  asm volatile("" ::: "memory");

  const bool bad = __builtin_amdgcn_readfirstlane(__hip_atomic_load(tmo, __ATOMIC_RELAXED, __HIP_MEMORY_SCOPE_AGENT)) == call;
  const float poison = bad ? __builtin_nanf("") : 0.0f;
  const int col = jt * 128 + wn * 64 + (lane >> 4) * 4;
  float* o = P.out + (size_t)(srcr * MSH + half * 256 + wm * 128 + grp * 64 + (lane & 15)) * NSH + col;
  const int er = wm + 2 * grp;
  float* oe = P.out + (size_t)(rank * MSH + (e0 + (er < ne ? er : 0)) * 16 + (lane & 15)) * NSH + col;
  auto put = [&](float* dst, f32x4 v) {
    f32x4 r;
#pragma unroll
    for (int e = 0; e < 4; ++e) r[e] = gelu_tanh(v[e] + poison);
    *(f32x4*)dst = r;
  };
  {
    auto finish = [&](auto G) {
      constexpr int g = decltype(G)::value;
      LAS char* mine = lds + wave * 16384 + lane * 16;
      const LAS char* theirs = lds + (wave ^ 4) * 16384 + lane * 16;
#pragma unroll
      for (int q = 0; q < 3; ++q) {
#pragma unroll
        for (int j = 0; j < 4; ++j) *(LAS f32x4*)(mine + (q * 5 + j) * 1024) = acc[(1 - g) * 4 + q][j];
        *(LAS f32x4*)(mine + (q * 5 + 4) * 1024) = acce[1 - g][q];
      }
      __builtin_amdgcn_s_waitcnt(0xC07F);
      __builtin_amdgcn_s_barrier();
      asm volatile("" ::: "memory");
#pragma unroll
      for (int q = 0; q < 3; ++q) {
#pragma unroll
        for (int j = 0; j < 4; ++j) put(o + (size_t)(q * 16) * NSH + j * 16, acc[g * 4 + q][j] + *(const LAS f32x4*)(theirs + (q * 5 + j) * 1024));
        const f32x4 v = acce[g][q] + *(const LAS f32x4*)(theirs + (q * 5 + 4) * 1024);
        if (er < ne) put(oe + q * 16, v);
      }
      __builtin_amdgcn_s_waitcnt(0xC07F);
      __builtin_amdgcn_s_barrier();
      asm volatile("" ::: "memory");
#pragma unroll
      for (int j = 0; j < 4; ++j) *(LAS f32x4*)(mine + j * 1024) = acc[(1 - g) * 4 + 3][j];
      *(LAS f32x4*)(mine + 4 * 1024) = acce[1 - g][3];
      __builtin_amdgcn_s_waitcnt(0xC07F);
      __builtin_amdgcn_s_barrier();
      asm volatile("" ::: "memory");
#pragma unroll
      for (int j = 0; j < 4; ++j) put(o + (size_t)(3 * 16) * NSH + j * 16, acc[g * 4 + 3][j] + *(const LAS f32x4*)(theirs + j * 1024));
      const f32x4 v = acce[g][3] + *(const LAS f32x4*)(theirs + 4 * 1024);
      if (er < ne) put(oe + 3 * 16, v);
    };
    if (grp == 0) finish(std::integral_constant<int, 0>{}); else finish(std::integral_constant<int, 1>{});
  }
}

extern "C" void kernel_launch(int rank, int nranks, ncclComm_t comm,
                              hipStream_t stream,
                              void* const* d_in, const int* in_sizes, int n_in,
                              void* d_out, int out_size,
                              const int* in_dtypes, int out_dtype,
                              void* const* d_out_all,
                              void* const* d_ws_all, size_t ws_bytes) {
  (void)comm; (void)in_sizes; (void)n_in; (void)out_size; (void)in_dtypes; (void)out_dtype; (void)d_out_all;
  if (nranks != NR || ws_bytes < WS_NEED || (unsigned)rank >= (unsigned)NR) {
    fprintf(stderr, "unsupported configuration (nranks %d, ws_bytes %zu)\n", nranks, ws_bytes);
    return;
  }
  Params P;
  for (int i = 0; i < NR; ++i) P.peer[i] = (char*)d_ws_all[(rank + 1 + i) % NR];
  P.x = (const float*)d_in[0];
  P.w = (const float*)d_in[1];
  P.out = (float*)d_out;
  P.rank = rank;
  ag_gemm_gelu_kernel<<<NWG, NT, 0, stream>>>(P);
  HIP_CHECK(hipGetLastError());
}
```
